```python
import numpy as np

kernel_name = "gpu_gemm_ar_m4096_k4096_n4096_bf16_q8"


ABSMAX_RTOL = 0.02

COMPUTE_DTYPE = "bfloat16"
EPILOGUE = "quant_int8"
IN_QUANT = None
OUT_QUANT = "int8"
IN_DTYPES = {"x": "bfloat16", "w_mat": "bfloat16"}
WORLD_SIZE = 8

def _to_bf16(x):
    u = np.ascontiguousarray(x, dtype=np.float32).view(np.uint32)
    return ((u + 0x7FFF + ((u >> 16) & 1)) & 0xFFFF0000).view(np.float32)

def setup_inputs():

    x = _to_bf16(np.random.randn(4096, 4096).astype(np.float32))
    w_mat = _to_bf16((np.random.randn(4096, 4096) / np.sqrt(4096)).astype(np.float32))
    return {"x": x, "w_mat": w_mat}

def _quant_dequant(y, amax):
    scale = amax / 127.0
    q = np.clip(np.round(y / scale), -127.0, 127.0)
    return (q * scale).astype(np.float32)
def _epilogue(y):
    return _quant_dequant(y, np.abs(y).max())

def reference(x, w_mat):
    y = (x @ w_mat).astype(np.float32)
    return _epilogue(y)
```

```cpp
#include <hip/hip_runtime.h>
#include <rccl/rccl.h>
#include <cstdio>
#include <cstdint>

namespace {

typedef __attribute__((ext_vector_type(8))) short bf16x8;
typedef __attribute__((ext_vector_type(4))) float f32x4;
typedef __attribute__((ext_vector_type(4))) unsigned u32x4;
typedef __attribute__((ext_vector_type(2))) unsigned u32x2;
typedef __attribute__((ext_vector_type(4))) short i16x4;
typedef unsigned long long u64;
#define LAS __attribute__((address_space(3)))

constexpr int WORLD = 8;
constexpr int MM = 4096, NN = 4096, KK = 4096;
constexpr int KS = KK / WORLD;
constexpr int KT = KS / 64;
constexpr int GJ = 4;
constexpr int MB = 2048, NB = 1024;
constexpr int RBS = MB / 128;
constexpr int CBS = NB / 128;
constexpr int NTILE = RBS * CBS;
constexpr size_t UNIT = 16384;

constexpr int NPUSH = 128;
constexpr int WPD = NPUSH / WORLD;
constexpr int UPD = KT * (RBS + CBS);
constexpr int GRID = NPUSH + NTILE;

constexpr size_t OFF_A = 0;
constexpr size_t OFF_B = OFF_A + (size_t)WORLD * KT * RBS * UNIT;
constexpr size_t OFF_C = OFF_B + (size_t)WORLD * KT * CBS * UNIT;
constexpr size_t C_FLAGA = 0;
constexpr size_t C_FLAGB = 4096;
constexpr size_t C_AMAX = 8192;
constexpr size_t C_DONE = 16384;
constexpr size_t C_CTR = 24576;
constexpr size_t C_TMO = 28672;
constexpr size_t WS_NEED = OFF_C + 32768;

constexpr int LDS_STAGE = 65536;
constexpr int LDS_BYTES = LDS_STAGE + 256;
constexpr unsigned long long DEADLINE = 20000000ull;

struct Params {
    const unsigned short* x;
    const unsigned short* w;
    char* ws[WORLD];
    signed char* out[WORLD];
    int rank;
};

__device__ __forceinline__ void drain() { asm volatile("s_waitcnt vmcnt(0)" ::: "memory"); }

__device__ __forceinline__ char* pick_ws(const Params& p, int d) {
    char* r = p.ws[0];
#pragma unroll
    for (int i = 1; i < WORLD; ++i) r = (d == i) ? p.ws[i] : r;
    return r;
}
__device__ __forceinline__ signed char* pick_out(const Params& p, int d) {
    signed char* r = p.out[0];
#pragma unroll
    for (int i = 1; i < WORLD; ++i) r = (d == i) ? p.out[i] : r;
    return r;
}

__device__ __forceinline__ bool wait_ge(const unsigned* flag, unsigned call, unsigned* tmo) {
    const unsigned long long t0 = __builtin_amdgcn_s_memrealtime();
    for (;;) {
        bool ok = true;
        if (flag) ok = __hip_atomic_load(flag, __ATOMIC_RELAXED, __HIP_MEMORY_SCOPE_SYSTEM) >= call;
        if (__all(ok)) return true;
        if (__hip_atomic_load(tmo, __ATOMIC_RELAXED, __HIP_MEMORY_SCOPE_AGENT) == call ||
            __builtin_amdgcn_s_memrealtime() - t0 > DEADLINE) {
            __hip_atomic_store(tmo, call, __ATOMIC_RELAXED, __HIP_MEMORY_SCOPE_AGENT);
            return false;
        }
        __builtin_amdgcn_s_sleep(2);
    }
}

__device__ __forceinline__ unsigned toff(unsigned row, unsigned ch) {
    return 256u * row + 16u * (ch ^ (((row & 3) << 2) | ((row >> 2) & 3)));
}

__device__ __forceinline__ void push_role(const Params& p, char* smem, unsigned call) {
    const int tid = threadIdx.x, lane = tid & 63, wv = tid >> 6;
    const int w = blockIdx.x;
    const int n = w >> 3;
    const int d = __builtin_amdgcn_readfirstlane((p.rank + 1 + ((w + n) & 7)) & 7);
    const int di = d >> 2, dj = d & 3;
    char* const dws = pick_ws(p, d);
    unsigned* const dflag = (unsigned*)(dws + OFF_C);
    for (int idx = n; idx < UPD; idx += WPD) {
        const int kt = idx / (RBS + CBS), r = idx % (RBS + CBS);
        const int k0 = kt * 64;
        u32x4 v[4];
        char* dst;
        unsigned* flag;
        if (r < RBS) {
            const int row0 = di * MB + r * 128;
#pragma unroll
            for (int q = 0; q < 4; ++q) {
                const int sub = wv * 4 + q;
                const int row = row0 + (sub >> 1) * 16 + (lane & 15);
                const int k = k0 + (sub & 1) * 32 + (lane >> 4) * 8;
                v[q] = *(const u32x4*)(p.x + (size_t)row * KS + k);
            }
            dst = dws + OFF_A + (size_t)((p.rank * KT + kt) * RBS + r) * UNIT;
            flag = dflag + C_FLAGA / 4 + (kt * RBS + r) * WORLD + p.rank;
        } else {
            const int c = r - RBS;
            const int col0 = dj * NB + c * 128;
#pragma unroll
            for (int q = 0; q < 4; ++q) {
                const int krow = q * 16 + (tid >> 4);
                v[q] = *(const u32x4*)(p.w + (size_t)(k0 + krow) * NN + col0 + (tid & 15) * 8);
            }
#pragma unroll
            for (int q = 0; q < 4; ++q) *(u32x4*)(smem + toff(q * 16 + (tid >> 4), tid & 15)) = v[q];
            __syncthreads();
#pragma unroll
            for (int q = 0; q < 4; ++q) {
                const int sub = wv * 4 + q, cb = sub >> 1, ks = sub & 1;
                const unsigned g = lane >> 4, qq = (lane & 15) >> 2, pp = lane & 3;
                const unsigned a0 = toff(32 * ks + 8 * g + qq, 2 * cb + (pp >> 1)) + 8 * (pp & 1);
                const unsigned a1 = toff(32 * ks + 8 * g + 4 + qq, 2 * cb + (pp >> 1)) + 8 * (pp & 1);
                const i16x4 t0 = __builtin_amdgcn_ds_read_tr16_b64_v4i16((LAS i16x4*)(smem + a0));
                const i16x4 t1 = __builtin_amdgcn_ds_read_tr16_b64_v4i16((LAS i16x4*)(smem + a1));
                union { struct { i16x4 a, b; } s; u32x4 u; } cv; cv.s.a = t0; cv.s.b = t1;
                v[q] = cv.u;
            }
            dst = dws + OFF_B + (size_t)((p.rank * KT + kt) * CBS + c) * UNIT;
            flag = dflag + C_FLAGB / 4 + (kt * CBS + c) * WORLD + p.rank;
        }
#pragma unroll
        for (int q = 0; q < 4; ++q) *(u32x4*)(dst + (wv * 4 + q) * 1024 + lane * 16) = v[q];
        drain();
        __syncthreads();
        if (tid == 0) {
            __builtin_amdgcn_fence(__ATOMIC_RELEASE, "");
            drain();
            __hip_atomic_store(flag, call, __ATOMIC_RELAXED, __HIP_MEMORY_SCOPE_SYSTEM);
        }
    }
}

__device__ __forceinline__ void gemm_role(const Params& p, char* smem, unsigned call) {
    const int tid = threadIdx.x, lane = tid & 63, wv = tid >> 6;
    const int g = blockIdx.x - NPUSH;
    const int RB = ((g & 7) >> 1) * 4 + ((g >> 3) >> 2);
    const int CB = (g & 1) * 4 + ((g >> 3) & 3);
    const int wr = wv >> 1, wc = wv & 1;
    char* const lws = p.ws[p.rank];
    unsigned* const lctl = (unsigned*)(lws + OFF_C);
    unsigned* const tmo = lctl + C_TMO / 4;
    const char* const srcA = lws + OFF_A + (size_t)RB * UNIT + tid * 16;
    const char* const srcB = lws + OFF_B + (size_t)CB * UNIT + tid * 16;

    f32x4 acc[4][4];
#pragma unroll
    for (int i = 0; i < 4; ++i)
#pragma unroll
        for (int j = 0; j < 4; ++j) acc[i][j] = (f32x4){0.f, 0.f, 0.f, 0.f};

    auto stage = [&](int t, int buf) {
        const int kt = t >> 3, s = (p.rank + t) & 7;
        const char* a = srcA + (size_t)(s * KT + kt) * RBS * UNIT;
        const char* b = srcB + (size_t)(s * KT + kt) * CBS * UNIT;
        char* la = smem + buf * 32768 + tid * 16;
#pragma unroll
        for (int q = 0; q < 4; ++q)
            __builtin_amdgcn_global_load_lds((const unsigned*)(a + q * 4096), (LAS unsigned*)(la + q * 4096), 16, 0, 0);
#pragma unroll
        for (int q = 0; q < 4; ++q)
            __builtin_amdgcn_global_load_lds((const unsigned*)(b + q * 4096), (LAS unsigned*)(la + 16384 + q * 4096), 16, 0, 0);
    };
    auto compute = [&](int buf) {
        const char* la = smem + buf * 32768 + lane * 16;
#pragma unroll
        for (int ks = 0; ks < 2; ++ks) {
            bf16x8 af[4], bfr[4];
#pragma unroll
            for (int i = 0; i < 4; ++i) af[i] = *(const bf16x8*)(la + ((wr * 4 + i) * 2 + ks) * 1024);
#pragma unroll
            for (int j = 0; j < 4; ++j) bfr[j] = *(const bf16x8*)(la + 16384 + ((wc * 4 + j) * 2 + ks) * 1024);
#pragma unroll
            for (int i = 0; i < 4; ++i)
#pragma unroll
                for (int j = 0; j < 4; ++j)
                    acc[i][j] = __builtin_amdgcn_mfma_f32_16x16x32_bf16(bfr[j], af[i], acc[i][j], 0, 0, 0);
        }
    };

    for (int t = 0; t < 64; ++t) {
        if ((t & 7) == 0) {
            if (wv == 0) {
                const int kt = t >> 3;
                const unsigned* f = nullptr;
                if (lane < 8) f = lctl + C_FLAGA / 4 + (kt * RBS + RB) * WORLD + lane;
                else if (lane < 16) f = lctl + C_FLAGB / 4 + (kt * CBS + CB) * WORLD + (lane - 8);
                wait_ge(f, call, tmo);
                __builtin_amdgcn_fence(__ATOMIC_ACQUIRE, "");
                drain();
            }
            __syncthreads();
        }
        stage(t, t & 1);
        if (t > 0) compute((t - 1) & 1);
        drain();
        __syncthreads();
    }
    compute(1);

    float m = 0.f;
#pragma unroll
    for (int i = 0; i < 4; ++i)
#pragma unroll
        for (int j = 0; j < 4; ++j)
#pragma unroll
            for (int e = 0; e < 4; ++e) m = fmaxf(m, fabsf(acc[i][j][e]));
#pragma unroll
    for (int o = 32; o > 0; o >>= 1) m = fmaxf(m, __shfl_xor(m, o));
    float* red = (float*)(smem + LDS_STAGE + 64);
    if (lane == 0) red[wv] = m;
    __syncthreads();
    if (tid < WORLD) {
        const float tm = fmaxf(fmaxf(red[0], red[1]), fmaxf(red[2], red[3]));
        const u64 gr = ((u64)call << 32) | (u64)__float_as_uint(tm);
        __hip_atomic_store((u64*)(p.ws[tid] + OFF_C + C_AMAX) + p.rank * NTILE + g, gr, __ATOMIC_RELAXED,
                           __HIP_MEMORY_SCOPE_SYSTEM);
    }
    {
        const u64* gp = (const u64*)(lws + OFF_C + C_AMAX) + tid * 4;
        const unsigned long long t0 = __builtin_amdgcn_s_memrealtime();
        float gm;
        for (;;) {
            bool ok = true;
            gm = 0.f;
#pragma unroll
            for (int e = 0; e < 4; ++e) {
                const u64 x = __hip_atomic_load(gp + e, __ATOMIC_RELAXED, __HIP_MEMORY_SCOPE_SYSTEM);
                ok = ok && ((unsigned)(x >> 32) == call);
                gm = fmaxf(gm, __uint_as_float((unsigned)x));
            }
            if (__all(ok)) break;
            if (__hip_atomic_load(tmo, __ATOMIC_RELAXED, __HIP_MEMORY_SCOPE_AGENT) == call ||
                __builtin_amdgcn_s_memrealtime() - t0 > DEADLINE) {
                __hip_atomic_store(tmo, call, __ATOMIC_RELAXED, __HIP_MEMORY_SCOPE_AGENT);
                break;
            }
            __builtin_amdgcn_s_sleep(1);
        }
#pragma unroll
        for (int o = 32; o > 0; o >>= 1) gm = fmaxf(gm, __shfl_xor(gm, o));
        __syncthreads();
        if (lane == 0) red[wv] = gm;
        __syncthreads();
    }
    const float amax = fmaxf(fmaxf(red[0], red[1]), fmaxf(red[2], red[3]));
    const bool bad = __builtin_amdgcn_readfirstlane(__hip_atomic_load(tmo, __ATOMIC_RELAXED, __HIP_MEMORY_SCOPE_AGENT)) == call;
    const float scale = amax / 127.0f;
    const float inv = scale > 0.f ? 1.0f / scale : 0.f;

    constexpr int QS = 144;
#pragma unroll
    for (int i = 0; i < 4; ++i) {
        const int row = wr * 64 + i * 16 + (lane & 15);
#pragma unroll
        for (int j = 0; j < 4; ++j) {
            unsigned pk = 0;
#pragma unroll
            for (int e = 0; e < 4; ++e) {
                float q = rintf(acc[i][j][e] * inv);
                q = fminf(fmaxf(q, -127.f), 127.f);
                pk |= ((unsigned)(int)q & 0xffu) << (8 * e);
            }
            if (bad) pk = 0x80808080u;
            *(unsigned*)(smem + row * QS + wc * 64 + j * 16 + (lane >> 4) * 4) = pk;
        }
    }
    __syncthreads();
    u32x4 v[4];
#pragma unroll
    for (int q = 0; q < 4; ++q) v[q] = *(const u32x4*)(smem + ((tid >> 3) + 32 * q) * QS + (tid & 7) * 16);
    const size_t obase = (size_t)((p.rank >> 2) * MB + RB * 128 + (tid >> 3)) * NN + (p.rank & 3) * NB + CB * 128 + (tid & 7) * 16;
#pragma unroll
    for (int dd = 0; dd < WORLD; ++dd) {
        const int d = __builtin_amdgcn_readfirstlane((p.rank + 1 + dd + g) & 7);
        if (bad && d != p.rank) continue;
        signed char* o = pick_out(p, d) + obase;
#pragma unroll
        for (int q = 0; q < 4; ++q) *(u32x4*)(o + (size_t)q * 32 * NN) = v[q];
    }
    drain();
    __syncthreads();
    if (wv == 0) {
        __builtin_amdgcn_fence(__ATOMIC_RELEASE, "");
        drain();
        if (lane < WORLD && lane != p.rank)
            __hip_atomic_store((unsigned*)(p.ws[lane] + OFF_C + C_DONE) + p.rank * NTILE + g, call, __ATOMIC_RELAXED,
                               __HIP_MEMORY_SCOPE_SYSTEM);
        const unsigned* f = (lane < WORLD && lane != p.rank) ? lctl + C_DONE / 4 + lane * NTILE + g : nullptr;
        if (!wait_ge(f, call, tmo) && lane == 0) p.out[p.rank][obase] = (signed char)-128;
    }
}

__global__ __launch_bounds__(256) void fused_gemm_ar_q8(Params p) {
    extern __shared__ __attribute__((aligned(16))) char smem[];
    unsigned* const sh = (unsigned*)(smem + LDS_STAGE);
    if (threadIdx.x == 0) {
        unsigned* ctr = (unsigned*)(p.ws[p.rank] + OFF_C + C_CTR) + blockIdx.x;
        const unsigned c = __hip_atomic_load(ctr, __ATOMIC_RELAXED, __HIP_MEMORY_SCOPE_AGENT) + 1u;
        __hip_atomic_store(ctr, c, __ATOMIC_RELAXED, __HIP_MEMORY_SCOPE_AGENT);
        sh[0] = c;
    }
    __syncthreads();
    const unsigned call = sh[0];
    if (blockIdx.x < NPUSH) push_role(p, smem, call);
    else gemm_role(p, smem, call);
}

}

extern "C" void kernel_launch(int rank, int nranks, ncclComm_t comm,
                              hipStream_t stream,
                              void* const* d_in, const int* in_sizes, int n_in,
                              void* d_out, int out_size,
                              const int* in_dtypes, int out_dtype,
                              void* const* d_out_all,
                              void* const* d_ws_all, size_t ws_bytes) {
    (void)comm; (void)in_sizes; (void)n_in; (void)d_out; (void)out_size; (void)in_dtypes; (void)out_dtype;
    if (nranks != WORLD || ws_bytes < WS_NEED) return;
    static bool init = false;
    if (!init) {
        (void)hipFuncSetAttribute((const void*)fused_gemm_ar_q8, hipFuncAttributeMaxDynamicSharedMemorySize, LDS_BYTES);
        init = true;
    }
    Params p;
    p.x = (const unsigned short*)d_in[0];
    p.w = (const unsigned short*)d_in[1];
    for (int i = 0; i < WORLD; ++i) { p.ws[i] = (char*)d_ws_all[i]; p.out[i] = (signed char*)d_out_all[i]; }
    p.rank = rank;
    hipLaunchKernelGGL(fused_gemm_ar_q8, dim3(GRID), dim3(256), LDS_BYTES, stream, p);
}
```

```cpp
#include <hip/hip_runtime.h>
#include <rccl/rccl.h>
#include <cstdio>
#include <cstdint>

namespace {

typedef __attribute__((ext_vector_type(8))) short bf16x8;
typedef __attribute__((ext_vector_type(4))) float f32x4;
typedef __attribute__((ext_vector_type(4))) unsigned u32x4;
typedef __attribute__((ext_vector_type(2))) unsigned u32x2;
typedef __attribute__((ext_vector_type(4))) short i16x4;
typedef unsigned long long u64;
#define LAS __attribute__((address_space(3)))

constexpr int WORLD = 8;
constexpr int MM = 4096, NN = 4096, KK = 4096;
constexpr int KS = KK / WORLD;
constexpr int KT = KS / 64;
constexpr int GJ = 4;
constexpr int MB = 2048, NB = 1024;
constexpr int RBS = MB / 128;
constexpr int CBS = NB / 128;
constexpr int NTILE = RBS * CBS;
constexpr size_t UNIT = 16384;

#define NPUSH_WG 48
#define PUSH_WAVES 4
constexpr int PW = PUSH_WAVES;
constexpr int NT = 512;
constexpr int NPUSH = NPUSH_WG;
constexpr int WPD = NPUSH / WORLD;
constexpr int UPD = KT * (RBS + CBS);
constexpr int GRID = NPUSH + NTILE;

constexpr size_t OFF_A = 0;
constexpr size_t OFF_B = OFF_A + (size_t)WORLD * KT * RBS * UNIT;
constexpr size_t OFF_C = OFF_B + (size_t)WORLD * KT * CBS * UNIT;
constexpr size_t C_FLAGA = 0;
constexpr size_t C_FLAGB = 4096;
constexpr size_t C_AMAX = 8192;
constexpr size_t C_DONE = 16384;
constexpr size_t C_CTR = 24576;
constexpr size_t C_TMO = 28672;
constexpr size_t WS_NEED = OFF_C + 32768;

constexpr int LDS_STAGE = 131072;
constexpr int LDS_BYTES = LDS_STAGE + 256;
#define GLDS_AUX 17
constexpr unsigned long long DEADLINE = 20000000ull;

struct Params {
    const unsigned short* x;
    const unsigned short* w;
    char* ws[WORLD];
    signed char* out[WORLD];
    int rank;
};

__device__ __forceinline__ void drain() { asm volatile("s_waitcnt vmcnt(0)" ::: "memory"); }

__device__ __forceinline__ void st16_wt(void* p, u32x4 v) {
    asm volatile("global_store_dwordx4 %0, %1, off sc0 sc1\n\ts_nop 1" :: "v"(p), "v"(v) : "memory");
}

__device__ __forceinline__ char* pick_ws(const Params& p, int d) {
    char* r = p.ws[0];
#pragma unroll
    for (int i = 1; i < WORLD; ++i) r = (d == i) ? p.ws[i] : r;
    return r;
}
__device__ __forceinline__ signed char* pick_out(const Params& p, int d) {
    signed char* r = p.out[0];
#pragma unroll
    for (int i = 1; i < WORLD; ++i) r = (d == i) ? p.out[i] : r;
    return r;
}

#define STAMP(i) do { } while (0)

__device__ __forceinline__ bool wait_ge(const unsigned* flag, unsigned call, unsigned* tmo) {
    const unsigned long long t0 = __builtin_amdgcn_s_memrealtime();
    for (;;) {
        bool ok = true;
        if (flag) ok = __hip_atomic_load(flag, __ATOMIC_RELAXED, __HIP_MEMORY_SCOPE_SYSTEM) >= call;
        if (__all(ok)) return true;
        if (__hip_atomic_load(tmo, __ATOMIC_RELAXED, __HIP_MEMORY_SCOPE_AGENT) == call ||
            __builtin_amdgcn_s_memrealtime() - t0 > DEADLINE) {
            __hip_atomic_store(tmo, call, __ATOMIC_RELAXED, __HIP_MEMORY_SCOPE_AGENT);
            return false;
        }
        __builtin_amdgcn_s_sleep(2);
    }
}

__device__ __forceinline__ unsigned toff(unsigned row, unsigned ch) {
    return 256u * row + 16u * (ch ^ (((row & 3) << 2) | ((row >> 2) & 3)));
}

__device__ __forceinline__ void push_role(const Params& p, char* smem, unsigned call) {
    const int tid = threadIdx.x, lane = tid & 63, wv = tid >> 6;
    const int w = blockIdx.x;
    if (wv >= PW) return;
    const int n = (w >> 3) * PW + wv;
    const int d = __builtin_amdgcn_readfirstlane((p.rank + 1 + ((w + (w >> 3)) & 7)) & 7);
    const int di = d >> 2, dj = d & 3;
    char* const dws = pick_ws(p, d);
    unsigned* const dflag = (unsigned*)(dws + OFF_C);
    char* const lds = smem + wv * 16384;
    auto load_raw = [&](int idx, u32x4 (&v)[16]) {
        const int kt = idx / (RBS + CBS), r = idx % (RBS + CBS);
        if (r < RBS) {
            const unsigned short* src = p.x + (size_t)(di * MB + r * 128 + (lane & 15)) * KS + kt * 64 + (lane >> 4) * 8;
#pragma unroll
            for (int q = 0; q < 16; ++q) v[q] = *(const u32x4*)(src + (size_t)(q >> 1) * 16 * KS + (q & 1) * 32);
        } else {
            const unsigned short* src = p.w + (size_t)(kt * 64 + (lane >> 4)) * NN + dj * NB + (r - RBS) * 128 + (lane & 15) * 8;
#pragma unroll
            for (int q = 0; q < 16; ++q) v[q] = *(const u32x4*)(src + (size_t)q * 4 * NN);
        }
    };
    u32x4 v[16], nx[16];
    if (n < UPD) load_raw(n, nx);
    for (int idx = n; idx < UPD; idx += WPD * PW) {
        const int kt = idx / (RBS + CBS), r = idx % (RBS + CBS);
        char* dst;
        unsigned* flag;
        if (r < RBS) {
#pragma unroll
            for (int q = 0; q < 16; ++q) v[q] = nx[q];
            dst = dws + OFF_A + (size_t)((p.rank * KT + kt) * RBS + r) * UNIT;
            flag = dflag + C_FLAGA / 4 + (kt * RBS + r) * WORLD + p.rank;
        } else {
            const int c = r - RBS;
#pragma unroll
            for (int q = 0; q < 16; ++q) *(u32x4*)(lds + toff(q * 4 + (lane >> 4), lane & 15)) = nx[q];
            const unsigned g = lane >> 4, qq = (lane & 15) >> 2, pp = lane & 3;
#pragma unroll
            for (int q = 0; q < 16; ++q) {
                const int cb = q >> 1, ks = q & 1;
                const unsigned a0 = toff(32 * ks + 8 * g + qq, 2 * cb + (pp >> 1)) + 8 * (pp & 1);
                const unsigned a1 = toff(32 * ks + 8 * g + 4 + qq, 2 * cb + (pp >> 1)) + 8 * (pp & 1);
                const i16x4 t0 = __builtin_amdgcn_ds_read_tr16_b64_v4i16((LAS i16x4*)(lds + a0));
                const i16x4 t1 = __builtin_amdgcn_ds_read_tr16_b64_v4i16((LAS i16x4*)(lds + a1));
                union { struct { i16x4 a, b; } s; u32x4 u; } cv; cv.s.a = t0; cv.s.b = t1;
                v[q] = cv.u;
            }
            dst = dws + OFF_B + (size_t)((p.rank * KT + kt) * CBS + c) * UNIT;
            flag = dflag + C_FLAGB / 4 + (kt * CBS + c) * WORLD + p.rank;
        }
#pragma unroll
        for (int q = 0; q < 16; ++q) st16_wt(dst + q * 1024 + lane * 16, v[q]);
        if (idx + WPD * PW < UPD) {
            load_raw(idx + WPD * PW, nx);
            asm volatile("s_waitcnt vmcnt(16)" ::: "memory");
        } else {
            drain();
        }
        if (lane == 0) __hip_atomic_store(flag, call, __ATOMIC_RELAXED, __HIP_MEMORY_SCOPE_SYSTEM);
    }
    STAMP(1);
}

__device__ __forceinline__ void gemm_role(const Params& p, char* smem, unsigned call) {
    const int tid = threadIdx.x, lane = tid & 63, wv = tid >> 6;
    const int g = blockIdx.x - NPUSH;
    const int RB = ((g & 7) >> 1) * 4 + ((g >> 3) >> 2);
    const int CB = (g & 1) * 4 + ((g >> 3) & 3);
    const int wr = wv >> 2, wc = wv & 3;
    char* const lws = p.ws[p.rank];
    unsigned* const lctl = (unsigned*)(lws + OFF_C);
    unsigned* const tmo = lctl + C_TMO / 4;
    const char* const srcA = lws + OFF_A + (size_t)RB * UNIT + tid * 16;
    const char* const srcB = lws + OFF_B + (size_t)CB * UNIT + tid * 16;

    f32x4 acc[4][2];
#pragma unroll
    for (int i = 0; i < 4; ++i)
#pragma unroll
        for (int j = 0; j < 2; ++j) acc[i][j] = (f32x4){0.f, 0.f, 0.f, 0.f};

    auto stage = [&](int t, int buf) {
        const int kt = t >> 3, s = (p.rank + t) & 7;
        const char* a = srcA + (size_t)(s * KT + kt) * RBS * UNIT;
        const char* b = srcB + (size_t)(s * KT + kt) * CBS * UNIT;
        char* la = smem + buf * 32768 + tid * 16;
#pragma unroll
        for (int q = 0; q < 2; ++q)
            __builtin_amdgcn_global_load_lds((const unsigned*)(a + q * 8192), (LAS unsigned*)(la + q * 8192), 16, 0, GLDS_AUX);
#pragma unroll
        for (int q = 0; q < 2; ++q)
            __builtin_amdgcn_global_load_lds((const unsigned*)(b + q * 8192), (LAS unsigned*)(la + 16384 + q * 8192), 16, 0, GLDS_AUX);
    };
    auto compute = [&](int buf) {
        const char* la = smem + buf * 32768 + lane * 16;
#pragma unroll
        for (int ks = 0; ks < 2; ++ks) {
            bf16x8 af[4], bfr[2];
#pragma unroll
            for (int i = 0; i < 4; ++i) af[i] = *(const bf16x8*)(la + ((wr * 4 + i) * 2 + ks) * 1024);
#pragma unroll
            for (int j = 0; j < 2; ++j) bfr[j] = *(const bf16x8*)(la + 16384 + ((wc * 2 + j) * 2 + ks) * 1024);
#pragma unroll
            for (int i = 0; i < 4; ++i)
#pragma unroll
                for (int j = 0; j < 2; ++j)
                    acc[i][j] = __builtin_amdgcn_mfma_f32_16x16x32_bf16(bfr[j], af[i], acc[i][j], 0, 0, 0);
        }
    };
    unsigned long long rmask = 0;
    auto wait_tile = [&](int t) {
        const int kt = t >> 3, s = (p.rank + t) & 7;
        if ((t & 7) == 0) rmask = 0;
        const unsigned long long need = (1ull << s) | (1ull << (8 + s));
        if ((rmask & need) == need) return;
        const unsigned* f = nullptr;
        if (lane < 8) f = lctl + C_FLAGA / 4 + (kt * RBS + RB) * WORLD + lane;
        else if (lane < 16) f = lctl + C_FLAGB / 4 + (kt * CBS + CB) * WORLD + (lane - 8);
        const unsigned long long t0 = __builtin_amdgcn_s_memrealtime();
        for (;;) {
            bool ok = false;
            if (f) ok = __hip_atomic_load(f, __ATOMIC_RELAXED, __HIP_MEMORY_SCOPE_SYSTEM) >= call;
            rmask = __ballot(ok);
            if ((rmask & need) == need) break;
            if (__hip_atomic_load(tmo, __ATOMIC_RELAXED, __HIP_MEMORY_SCOPE_AGENT) == call ||
                __builtin_amdgcn_s_memrealtime() - t0 > DEADLINE) {
                __hip_atomic_store(tmo, call, __ATOMIC_RELAXED, __HIP_MEMORY_SCOPE_AGENT);
                rmask = ~0ull;
                break;
            }
            __builtin_amdgcn_s_sleep(1);
        }
        __builtin_amdgcn_fence(__ATOMIC_ACQUIRE, "wavefront");
        if ((t & 7) == 7) STAMP(2 + kt);
    };
    wait_tile(0); stage(0, 0);
    wait_tile(1); stage(1, 1);
    wait_tile(2); stage(2, 2);
    for (int t = 0; t < 64; ++t) {
        if (t < 62) asm volatile("s_waitcnt vmcnt(8)" ::: "memory");
        else if (t == 62) asm volatile("s_waitcnt vmcnt(4)" ::: "memory");
        else asm volatile("s_waitcnt vmcnt(0)" ::: "memory");
        __builtin_amdgcn_s_barrier();
        if (t + 3 < 64) {
            wait_tile(t + 3);
            stage(t + 3, (t + 3) & 3);
        }
        compute(t & 3);
    }
    __builtin_amdgcn_s_barrier();
    STAMP(10);

    float m = 0.f;
#pragma unroll
    for (int i = 0; i < 4; ++i)
#pragma unroll
        for (int j = 0; j < 2; ++j)
#pragma unroll
            for (int e = 0; e < 4; ++e) m = fmaxf(m, fabsf(acc[i][j][e]));
#pragma unroll
    for (int o = 32; o > 0; o >>= 1) m = fmaxf(m, __shfl_xor(m, o));
    float* red = (float*)(smem + LDS_STAGE + 64);
    if (lane == 0) red[wv] = m;
    __syncthreads();
    if (tid < WORLD) {
        float tm = red[0];
#pragma unroll
        for (int i = 1; i < 8; ++i) tm = fmaxf(tm, red[i]);
        const u64 gr = ((u64)call << 32) | (u64)__float_as_uint(tm);
        __hip_atomic_store((u64*)(p.ws[tid] + OFF_C + C_AMAX) + p.rank * NTILE + g, gr, __ATOMIC_RELAXED,
                           __HIP_MEMORY_SCOPE_SYSTEM);
    }
    {
        const u64* gp = (const u64*)(lws + OFF_C + C_AMAX) + tid * 2;
        const unsigned long long t0 = __builtin_amdgcn_s_memrealtime();
        float gm;
        for (;;) {
            bool ok = true;
            gm = 0.f;
#pragma unroll
            for (int e = 0; e < 2; ++e) {
                const u64 x = __hip_atomic_load(gp + e, __ATOMIC_RELAXED, __HIP_MEMORY_SCOPE_SYSTEM);
                ok = ok && ((unsigned)(x >> 32) == call);
                gm = fmaxf(gm, __uint_as_float((unsigned)x));
            }
            if (__all(ok)) break;
            if (__hip_atomic_load(tmo, __ATOMIC_RELAXED, __HIP_MEMORY_SCOPE_AGENT) == call ||
                __builtin_amdgcn_s_memrealtime() - t0 > DEADLINE) {
                __hip_atomic_store(tmo, call, __ATOMIC_RELAXED, __HIP_MEMORY_SCOPE_AGENT);
                break;
            }
            __builtin_amdgcn_s_sleep(1);
        }
#pragma unroll
        for (int o = 32; o > 0; o >>= 1) gm = fmaxf(gm, __shfl_xor(gm, o));
        __syncthreads();
        if (lane == 0) red[wv] = gm;
        __syncthreads();
    }
    STAMP(11);
    float amax = red[0];
#pragma unroll
    for (int i = 1; i < 8; ++i) amax = fmaxf(amax, red[i]);
    const bool bad = __builtin_amdgcn_readfirstlane(__hip_atomic_load(tmo, __ATOMIC_RELAXED, __HIP_MEMORY_SCOPE_AGENT)) == call;
    const float scale = amax / 127.0f;
    const float inv = scale > 0.f ? 1.0f / scale : 0.f;

    constexpr int QS = 144;
#pragma unroll
    for (int i = 0; i < 4; ++i) {
        const int row = wr * 64 + i * 16 + (lane & 15);
#pragma unroll
        for (int j = 0; j < 2; ++j) {
            unsigned pk = 0;
#pragma unroll
            for (int e = 0; e < 4; ++e) {
                float q = rintf(acc[i][j][e] * inv);
                q = fminf(fmaxf(q, -127.f), 127.f);
                pk |= ((unsigned)(int)q & 0xffu) << (8 * e);
            }
            if (bad) pk = 0x80808080u;
            *(unsigned*)(smem + row * QS + wc * 32 + j * 16 + (lane >> 4) * 4) = pk;
        }
    }
    __syncthreads();
    {
        const int d0 = __builtin_amdgcn_readfirstlane((p.rank + 1 + (wv >> 1) + g) & 7);
        const int d1 = __builtin_amdgcn_readfirstlane((p.rank + 5 + (wv >> 1) + g) & 7);
        const int r0 = (wv & 1) * 64 + (lane >> 3);
        const size_t ob = (size_t)((p.rank >> 2) * MB + RB * 128 + r0) * NN + (p.rank & 3) * NB + CB * 128 + (lane & 7) * 16;
        signed char* o0 = pick_out(p, d0) + ob;
        signed char* o1 = pick_out(p, d1) + ob;
        const bool s0 = !bad || d0 == p.rank, s1 = !bad || d1 == p.rank;
#pragma unroll
        for (int q = 0; q < 8; ++q) {
            const u32x4 v = *(const u32x4*)(smem + (q * 8 + r0) * QS + (lane & 7) * 16);
            if (s0) st16_wt(o0 + (size_t)q * 8 * NN, v);
            if (s1) st16_wt(o1 + (size_t)q * 8 * NN, v);
        }
    }
    const size_t obase = (size_t)((p.rank >> 2) * MB + RB * 128) * NN + (p.rank & 3) * NB + CB * 128;
    drain();
    __syncthreads();
    STAMP(12);
    if (wv == 0) {
        if (lane < WORLD && lane != p.rank)
            __hip_atomic_store((unsigned*)(p.ws[lane] + OFF_C + C_DONE) + p.rank * NTILE + g, call, __ATOMIC_RELAXED,
                               __HIP_MEMORY_SCOPE_SYSTEM);
        const unsigned* f = (lane < WORLD && lane != p.rank) ? lctl + C_DONE / 4 + lane * NTILE + g : nullptr;
        if (!wait_ge(f, call, tmo) && lane == 0) p.out[p.rank][obase] = (signed char)-128;
    }
    STAMP(13);
}

__global__ __launch_bounds__(NT) void fused_gemm_ar_q8(Params p) {
    extern __shared__ __attribute__((aligned(16))) char smem[];
    unsigned* const sh = (unsigned*)(smem + LDS_STAGE);
    if (threadIdx.x == 0) {
        unsigned* ctr = (unsigned*)(p.ws[p.rank] + OFF_C + C_CTR) + blockIdx.x;
        const unsigned c = __hip_atomic_load(ctr, __ATOMIC_RELAXED, __HIP_MEMORY_SCOPE_AGENT) + 1u;
        __hip_atomic_store(ctr, c, __ATOMIC_RELAXED, __HIP_MEMORY_SCOPE_AGENT);
        sh[0] = c;
    }
    __syncthreads();
    const unsigned call = sh[0];
    STAMP(0);
    if (blockIdx.x < NPUSH) push_role(p, smem, call);
    else gemm_role(p, smem, call);
}

}

extern "C" void kernel_launch(int rank, int nranks, ncclComm_t comm,
                              hipStream_t stream,
                              void* const* d_in, const int* in_sizes, int n_in,
                              void* d_out, int out_size,
                              const int* in_dtypes, int out_dtype,
                              void* const* d_out_all,
                              void* const* d_ws_all, size_t ws_bytes) {
    (void)comm; (void)in_sizes; (void)n_in; (void)d_out; (void)out_size; (void)in_dtypes; (void)out_dtype;
    if (nranks != WORLD || ws_bytes < WS_NEED) return;
    static bool init = false;
    if (!init) {
        (void)hipFuncSetAttribute((const void*)fused_gemm_ar_q8, hipFuncAttributeMaxDynamicSharedMemorySize, LDS_BYTES);
        init = true;
    }
    Params p;
    p.x = (const unsigned short*)d_in[0];
    p.w = (const unsigned short*)d_in[1];
    for (int i = 0; i < WORLD; ++i) { p.ws[i] = (char*)d_ws_all[i]; p.out[i] = (signed char*)d_out_all[i]; }
    p.rank = rank;
    hipLaunchKernelGGL(fused_gemm_ar_q8, dim3(GRID), dim3(NT), LDS_BYTES, stream, p);
}
```

```cpp
#include <hip/hip_runtime.h>
#include <rccl/rccl.h>
#include <cstdio>
#include <cstdint>

namespace {

typedef __attribute__((ext_vector_type(8))) short bf16x8;
typedef __attribute__((ext_vector_type(4))) float f32x4;
typedef __attribute__((ext_vector_type(4))) unsigned u32x4;
typedef __attribute__((ext_vector_type(2))) unsigned u32x2;
typedef __attribute__((ext_vector_type(4))) short i16x4;
typedef unsigned long long u64;
#define LAS __attribute__((address_space(3)))

constexpr int WORLD = 8;
constexpr int MM = 4096, NN = 4096, KK = 4096;
constexpr int KS = KK / WORLD;
constexpr int KT = KS / 64;
constexpr int GJ = 4;
constexpr int MB = 2048, NB = 1024;
constexpr int RBS = MB / 128;
constexpr int CBS = NB / 128;
constexpr int NTILE = RBS * CBS;
constexpr size_t UNIT = 16384;

#define NPUSH_WG 48
#define PUSH_WAVES 4
constexpr int PW = PUSH_WAVES;
constexpr int NT = 512;
constexpr int NPUSH = NPUSH_WG;
constexpr int WPD = NPUSH / WORLD;
constexpr int UPD = KT * (RBS + CBS);
constexpr int GRID = NPUSH + NTILE;

constexpr size_t OFF_A = 0;
constexpr size_t OFF_B = OFF_A + (size_t)WORLD * KT * RBS * UNIT;
constexpr size_t OFF_C = OFF_B + (size_t)WORLD * KT * CBS * UNIT;
constexpr size_t C_FLAGA = 0;
constexpr size_t C_FLAGB = 4096;
constexpr size_t C_AMAX = 8192;
constexpr size_t C_DONE = 16384;
constexpr size_t C_CTR = 24576;
constexpr size_t C_TMO = 28672;
constexpr size_t WS_NEED = OFF_C + 32768;

constexpr int LDS_STAGE = 131072;
constexpr int LDS_BYTES = LDS_STAGE + 256;
#define GLDS_AUX 17
constexpr unsigned long long DEADLINE = 20000000ull;

struct Params {
    const unsigned short* x;
    const unsigned short* w;
    char* ws[WORLD];
    signed char* out[WORLD];
    int rank;
};

__device__ __forceinline__ void drain() { asm volatile("s_waitcnt vmcnt(0)" ::: "memory"); }

__device__ __forceinline__ void st16_wt(void* p, u32x4 v) {
    asm volatile("global_store_dwordx4 %0, %1, off sc0 sc1\n\ts_nop 1" :: "v"(p), "v"(v) : "memory");
}

__device__ __forceinline__ char* pick_ws(const Params& p, int d) {
    char* r = p.ws[0];
#pragma unroll
    for (int i = 1; i < WORLD; ++i) r = (d == i) ? p.ws[i] : r;
    return r;
}
__device__ __forceinline__ signed char* pick_out(const Params& p, int d) {
    signed char* r = p.out[0];
#pragma unroll
    for (int i = 1; i < WORLD; ++i) r = (d == i) ? p.out[i] : r;
    return r;
}

#define STAMP(i) do { } while (0)

__device__ __forceinline__ bool wait_ge(const unsigned* flag, unsigned call, unsigned* tmo) {
    const unsigned long long t0 = __builtin_amdgcn_s_memrealtime();
    for (;;) {
        bool ok = true;
        if (flag) ok = __hip_atomic_load(flag, __ATOMIC_RELAXED, __HIP_MEMORY_SCOPE_SYSTEM) >= call;
        if (__all(ok)) return true;
        if (__hip_atomic_load(tmo, __ATOMIC_RELAXED, __HIP_MEMORY_SCOPE_AGENT) == call ||
            __builtin_amdgcn_s_memrealtime() - t0 > DEADLINE) {
            __hip_atomic_store(tmo, call, __ATOMIC_RELAXED, __HIP_MEMORY_SCOPE_AGENT);
            return false;
        }
        __builtin_amdgcn_s_sleep(2);
    }
}

__device__ __forceinline__ unsigned toff(unsigned row, unsigned ch) {
    return 256u * row + 16u * (ch ^ (((row & 3) << 2) | ((row >> 2) & 3)));
}

__device__ __forceinline__ unsigned next_call(const Params& p, char* smem) {
    unsigned* const sh = (unsigned*)(smem + LDS_STAGE);
    if (threadIdx.x == 0) {
        unsigned* ctr = (unsigned*)(p.ws[p.rank] + OFF_C + C_CTR) + blockIdx.x;
        const unsigned c = __hip_atomic_load(ctr, __ATOMIC_RELAXED, __HIP_MEMORY_SCOPE_AGENT) + 1u;
        __hip_atomic_store(ctr, c, __ATOMIC_RELAXED, __HIP_MEMORY_SCOPE_AGENT);
        sh[0] = c;
    }
    __syncthreads();
    return sh[0];
}

__device__ __forceinline__ void push_role(const Params& p, char* smem) {
    const int tid = threadIdx.x, lane = tid & 63, wv = tid >> 6;
    const int w = blockIdx.x;
    if (wv >= PW) return;
    const int n = (w >> 3) * PW + wv;
    const int d = __builtin_amdgcn_readfirstlane((p.rank + 1 + ((w + (w >> 3)) & 7)) & 7);
    const int di = d >> 2, dj = d & 3;
    char* const dws = pick_ws(p, d);
    unsigned* const dflag = (unsigned*)(dws + OFF_C);
    char* const lds = smem + wv * 16384;
    auto load_raw = [&](int idx, u32x4 (&v)[16]) {
        const int kt = idx / (RBS + CBS), r = idx % (RBS + CBS);
        if (r < RBS) {
            const unsigned short* src = p.x + (size_t)(di * MB + r * 128 + (lane & 15)) * KS + kt * 64 + (lane >> 4) * 8;
#pragma unroll
            for (int q = 0; q < 16; ++q) v[q] = *(const u32x4*)(src + (size_t)(q >> 1) * 16 * KS + (q & 1) * 32);
        } else {
            const unsigned short* src = p.w + (size_t)(kt * 64 + (lane >> 4)) * NN + dj * NB + (r - RBS) * 128 + (lane & 15) * 8;
#pragma unroll
            for (int q = 0; q < 16; ++q) v[q] = *(const u32x4*)(src + (size_t)q * 4 * NN);
        }
    };
    u32x4 v[16], nx[16];
    if (n < UPD) load_raw(n, nx);
    const unsigned call = next_call(p, smem);
    STAMP(0);
    for (int idx = n; idx < UPD; idx += WPD * PW) {
        const int kt = idx / (RBS + CBS), r = idx % (RBS + CBS);
        char* dst;
        unsigned* flag;
        if (r < RBS) {
#pragma unroll
            for (int q = 0; q < 16; ++q) v[q] = nx[q];
            dst = dws + OFF_A + (size_t)((p.rank * KT + kt) * RBS + r) * UNIT;
            flag = dflag + C_FLAGA / 4 + (kt * RBS + r) * WORLD + p.rank;
        } else {
            const int c = r - RBS;
#pragma unroll
            for (int q = 0; q < 16; ++q) *(u32x4*)(lds + toff(q * 4 + (lane >> 4), lane & 15)) = nx[q];
            const unsigned g = lane >> 4, qq = (lane & 15) >> 2, pp = lane & 3;
#pragma unroll
            for (int q = 0; q < 16; ++q) {
                const int cb = q >> 1, ks = q & 1;
                const unsigned a0 = toff(32 * ks + 8 * g + qq, 2 * cb + (pp >> 1)) + 8 * (pp & 1);
                const unsigned a1 = toff(32 * ks + 8 * g + 4 + qq, 2 * cb + (pp >> 1)) + 8 * (pp & 1);
                const i16x4 t0 = __builtin_amdgcn_ds_read_tr16_b64_v4i16((LAS i16x4*)(lds + a0));
                const i16x4 t1 = __builtin_amdgcn_ds_read_tr16_b64_v4i16((LAS i16x4*)(lds + a1));
                union { struct { i16x4 a, b; } s; u32x4 u; } cv; cv.s.a = t0; cv.s.b = t1;
                v[q] = cv.u;
            }
            dst = dws + OFF_B + (size_t)((p.rank * KT + kt) * CBS + c) * UNIT;
            flag = dflag + C_FLAGB / 4 + (kt * CBS + c) * WORLD + p.rank;
        }
#pragma unroll
        for (int q = 0; q < 16; ++q) st16_wt(dst + q * 1024 + lane * 16, v[q]);
        if (idx + WPD * PW < UPD) {
            load_raw(idx + WPD * PW, nx);
            asm volatile("s_waitcnt vmcnt(16)" ::: "memory");
        } else {
            drain();
        }
        if (lane == 0) __hip_atomic_store(flag, call, __ATOMIC_RELAXED, __HIP_MEMORY_SCOPE_SYSTEM);
    }
    STAMP(1);
}

__device__ __forceinline__ void gemm_role(const Params& p, char* smem, unsigned call) {
    const int tid = threadIdx.x, lane = tid & 63, wv = tid >> 6;
    const int g = blockIdx.x - NPUSH;
    const int RB = ((g & 7) >> 1) * 4 + ((g >> 3) >> 2);
    const int CB = (g & 1) * 4 + ((g >> 3) & 3);
    const int wr = wv >> 2, wc = wv & 3;
    char* const lws = p.ws[p.rank];
    unsigned* const lctl = (unsigned*)(lws + OFF_C);
    unsigned* const tmo = lctl + C_TMO / 4;
    const char* const srcA = lws + OFF_A + (size_t)RB * UNIT + tid * 16;
    const char* const srcB = lws + OFF_B + (size_t)CB * UNIT + tid * 16;

    f32x4 acc[4][2];
#pragma unroll
    for (int i = 0; i < 4; ++i)
#pragma unroll
        for (int j = 0; j < 2; ++j) acc[i][j] = (f32x4){0.f, 0.f, 0.f, 0.f};

    auto stage = [&](int t, int buf) {
        const int kt = t >> 3, s = (p.rank + t) & 7;
        const char* a = srcA + (size_t)(s * KT + kt) * RBS * UNIT;
        const char* b = srcB + (size_t)(s * KT + kt) * CBS * UNIT;
        char* la = smem + buf * 32768 + tid * 16;
#pragma unroll
        for (int q = 0; q < 2; ++q)
            __builtin_amdgcn_global_load_lds((const unsigned*)(a + q * 8192), (LAS unsigned*)(la + q * 8192), 16, 0, GLDS_AUX);
#pragma unroll
        for (int q = 0; q < 2; ++q)
            __builtin_amdgcn_global_load_lds((const unsigned*)(b + q * 8192), (LAS unsigned*)(la + 16384 + q * 8192), 16, 0, GLDS_AUX);
    };
    auto compute = [&](int buf) {
        const char* la = smem + buf * 32768 + lane * 16;
#pragma unroll
        for (int ks = 0; ks < 2; ++ks) {
            bf16x8 af[4], bfr[2];
#pragma unroll
            for (int i = 0; i < 4; ++i) af[i] = *(const bf16x8*)(la + ((wr * 4 + i) * 2 + ks) * 1024);
#pragma unroll
            for (int j = 0; j < 2; ++j) bfr[j] = *(const bf16x8*)(la + 16384 + ((wc * 2 + j) * 2 + ks) * 1024);
#pragma unroll
            for (int i = 0; i < 4; ++i)
#pragma unroll
                for (int j = 0; j < 2; ++j)
                    acc[i][j] = __builtin_amdgcn_mfma_f32_16x16x32_bf16(bfr[j], af[i], acc[i][j], 0, 0, 0);
        }
    };
    unsigned long long rmask = 0;
    auto wait_tile = [&](int t) {
        const int kt = t >> 3, s = (p.rank + t) & 7;
        if ((t & 7) == 0) rmask = 0;
        const unsigned long long need = (1ull << s) | (1ull << (8 + s));
        if ((rmask & need) == need) return;
        const unsigned* f = nullptr;
        if (lane < 8) f = lctl + C_FLAGA / 4 + (kt * RBS + RB) * WORLD + lane;
        else if (lane < 16) f = lctl + C_FLAGB / 4 + (kt * CBS + CB) * WORLD + (lane - 8);
        const unsigned long long t0 = __builtin_amdgcn_s_memrealtime();
        for (;;) {
            bool ok = false;
            if (f) ok = __hip_atomic_load(f, __ATOMIC_RELAXED, __HIP_MEMORY_SCOPE_SYSTEM) >= call;
            rmask = __ballot(ok);
            if ((rmask & need) == need) break;
            if (__hip_atomic_load(tmo, __ATOMIC_RELAXED, __HIP_MEMORY_SCOPE_AGENT) == call ||
                __builtin_amdgcn_s_memrealtime() - t0 > DEADLINE) {
                __hip_atomic_store(tmo, call, __ATOMIC_RELAXED, __HIP_MEMORY_SCOPE_AGENT);
                rmask = ~0ull;
                break;
            }
            __builtin_amdgcn_s_sleep(1);
        }
        __builtin_amdgcn_fence(__ATOMIC_ACQUIRE, "wavefront");
        if ((t & 7) == 7) STAMP(2 + kt);
    };
    wait_tile(0); stage(0, 0);
    wait_tile(1); stage(1, 1);
    wait_tile(2); stage(2, 2);
    for (int t = 0; t < 64; ++t) {
        if (t < 62) asm volatile("s_waitcnt vmcnt(8)" ::: "memory");
        else if (t == 62) asm volatile("s_waitcnt vmcnt(4)" ::: "memory");
        else asm volatile("s_waitcnt vmcnt(0)" ::: "memory");
        __builtin_amdgcn_s_barrier();
        if (t + 3 < 64) {
            wait_tile(t + 3);
            stage(t + 3, (t + 3) & 3);
        }
        compute(t & 3);
    }
    __builtin_amdgcn_s_barrier();
    STAMP(10);

    float m = 0.f;
#pragma unroll
    for (int i = 0; i < 4; ++i)
#pragma unroll
        for (int j = 0; j < 2; ++j)
#pragma unroll
            for (int e = 0; e < 4; ++e) m = fmaxf(m, fabsf(acc[i][j][e]));
#pragma unroll
    for (int o = 32; o > 0; o >>= 1) m = fmaxf(m, __shfl_xor(m, o));
    float* red = (float*)(smem + LDS_STAGE + 64);
    if (lane == 0) red[wv] = m;
    __syncthreads();
    if (tid < WORLD) {
        float tm = red[0];
#pragma unroll
        for (int i = 1; i < 8; ++i) tm = fmaxf(tm, red[i]);
        const u64 gr = ((u64)call << 32) | (u64)__float_as_uint(tm);
        __hip_atomic_store((u64*)(p.ws[tid] + OFF_C + C_AMAX) + p.rank * NTILE + g, gr, __ATOMIC_RELAXED,
                           __HIP_MEMORY_SCOPE_SYSTEM);
    }
    {
        const u64* gp = (const u64*)(lws + OFF_C + C_AMAX) + tid * 2;
        const unsigned long long t0 = __builtin_amdgcn_s_memrealtime();
        float gm;
        for (;;) {
            bool ok = true;
            gm = 0.f;
#pragma unroll
            for (int e = 0; e < 2; ++e) {
                const u64 x = __hip_atomic_load(gp + e, __ATOMIC_RELAXED, __HIP_MEMORY_SCOPE_SYSTEM);
                ok = ok && ((unsigned)(x >> 32) == call);
                gm = fmaxf(gm, __uint_as_float((unsigned)x));
            }
            if (__all(ok)) break;
            if (__hip_atomic_load(tmo, __ATOMIC_RELAXED, __HIP_MEMORY_SCOPE_AGENT) == call ||
                __builtin_amdgcn_s_memrealtime() - t0 > DEADLINE) {
                __hip_atomic_store(tmo, call, __ATOMIC_RELAXED, __HIP_MEMORY_SCOPE_AGENT);
                break;
            }
            __builtin_amdgcn_s_sleep(1);
        }
#pragma unroll
        for (int o = 32; o > 0; o >>= 1) gm = fmaxf(gm, __shfl_xor(gm, o));
        __syncthreads();
        if (lane == 0) red[wv] = gm;
        __syncthreads();
    }
    STAMP(11);
    float amax = red[0];
#pragma unroll
    for (int i = 1; i < 8; ++i) amax = fmaxf(amax, red[i]);
    const bool bad = __builtin_amdgcn_readfirstlane(__hip_atomic_load(tmo, __ATOMIC_RELAXED, __HIP_MEMORY_SCOPE_AGENT)) == call;
    const float scale = amax / 127.0f;
    const float inv = scale > 0.f ? 1.0f / scale : 0.f;

    constexpr int QS = 144;
#pragma unroll
    for (int i = 0; i < 4; ++i) {
        const int row = wr * 64 + i * 16 + (lane & 15);
#pragma unroll
        for (int j = 0; j < 2; ++j) {
            unsigned pk = 0;
#pragma unroll
            for (int e = 0; e < 4; ++e) {
                float q = rintf(acc[i][j][e] * inv);
                q = fminf(fmaxf(q, -127.f), 127.f);
                pk |= ((unsigned)(int)q & 0xffu) << (8 * e);
            }
            if (bad) pk = 0x80808080u;
            *(unsigned*)(smem + row * QS + wc * 32 + j * 16 + (lane >> 4) * 4) = pk;
        }
    }
    __syncthreads();
    {
        const int d0 = __builtin_amdgcn_readfirstlane((p.rank + 1 + (wv >> 1) + g) & 7);
        const int d1 = __builtin_amdgcn_readfirstlane((p.rank + 5 + (wv >> 1) + g) & 7);
        const int r0 = (wv & 1) * 64 + (lane >> 3);
        const size_t ob = (size_t)((p.rank >> 2) * MB + RB * 128 + r0) * NN + (p.rank & 3) * NB + CB * 128 + (lane & 7) * 16;
        signed char* o0 = pick_out(p, d0) + ob;
        signed char* o1 = pick_out(p, d1) + ob;
        const bool s0 = !bad || d0 == p.rank, s1 = !bad || d1 == p.rank;
#pragma unroll
        for (int q = 0; q < 8; ++q) {
            const u32x4 v = *(const u32x4*)(smem + (q * 8 + r0) * QS + (lane & 7) * 16);
            if (s0) st16_wt(o0 + (size_t)q * 8 * NN, v);
            if (s1) st16_wt(o1 + (size_t)q * 8 * NN, v);
        }
    }
    const size_t obase = (size_t)((p.rank >> 2) * MB + RB * 128) * NN + (p.rank & 3) * NB + CB * 128;
    drain();
    __syncthreads();
    STAMP(12);
    if (wv == 0) {
        if (lane < WORLD && lane != p.rank)
            __hip_atomic_store((unsigned*)(p.ws[lane] + OFF_C + C_DONE) + p.rank * NTILE + g, call, __ATOMIC_RELAXED,
                               __HIP_MEMORY_SCOPE_SYSTEM);
        const unsigned* f = (lane < WORLD && lane != p.rank) ? lctl + C_DONE / 4 + lane * NTILE + g : nullptr;
        if (!wait_ge(f, call, tmo) && lane == 0) p.out[p.rank][obase] = (signed char)-128;
    }
    STAMP(13);
}

__global__ __launch_bounds__(NT) void fused_gemm_ar_q8(Params p) {
    extern __shared__ __attribute__((aligned(16))) char smem[];
    if (blockIdx.x < NPUSH) {
        push_role(p, smem);
    } else {
        const unsigned call = next_call(p, smem);
        STAMP(0);
        gemm_role(p, smem, call);
    }
}

}

extern "C" void kernel_launch(int rank, int nranks, ncclComm_t comm,
                              hipStream_t stream,
                              void* const* d_in, const int* in_sizes, int n_in,
                              void* d_out, int out_size,
                              const int* in_dtypes, int out_dtype,
                              void* const* d_out_all,
                              void* const* d_ws_all, size_t ws_bytes) {
    (void)comm; (void)in_sizes; (void)n_in; (void)d_out; (void)out_size; (void)in_dtypes; (void)out_dtype;
    if (nranks != WORLD || ws_bytes < WS_NEED) return;
    static bool init = false;
    if (!init) {
        (void)hipFuncSetAttribute((const void*)fused_gemm_ar_q8, hipFuncAttributeMaxDynamicSharedMemorySize, LDS_BYTES);
        init = true;
    }
    Params p;
    p.x = (const unsigned short*)d_in[0];
    p.w = (const unsigned short*)d_in[1];
    for (int i = 0; i < WORLD; ++i) { p.ws[i] = (char*)d_ws_all[i]; p.out[i] = (signed char*)d_out_all[i]; }
    p.rank = rank;
    hipLaunchKernelGGL(fused_gemm_ar_q8, dim3(GRID), dim3(NT), LDS_BYTES, stream, p);
}
```
